```python
import numpy as np

kernel_name = "gpu_ag_gemm_m4096_k4096_n32768_bf16_relu"


ABSMAX_RTOL = 0.008

COMPUTE_DTYPE = "bfloat16"
EPILOGUE = "relu"
IN_QUANT = None
OUT_QUANT = None
IN_DTYPES = {"x": "bfloat16", "w_mat": "bfloat16"}
WORLD_SIZE = 8

def _to_bf16(x):
    u = np.ascontiguousarray(x, dtype=np.float32).view(np.uint32)
    return ((u + 0x7FFF + ((u >> 16) & 1)) & 0xFFFF0000).view(np.float32)

def setup_inputs():

    x = _to_bf16(np.random.randn(4096, 4096).astype(np.float32))
    w_mat = _to_bf16((np.random.randn(4096, 32768) / np.sqrt(4096)).astype(np.float32))
    return {"x": x, "w_mat": w_mat}

def _epilogue(y):
    return np.maximum(y, 0.0).astype(np.float32)

def reference(x, w_mat):
    y = (x @ w_mat).astype(np.float32)
    return _epilogue(y)
```

```cpp
#include <hip/hip_runtime.h>
#include <rccl/rccl.h>
#include <cstdio>

#define HIP_CHECK(expr) do { hipError_t _e = (expr); if (_e != hipSuccess) { \
    fprintf(stderr, "HIP error %s at %s:%d\n", hipGetErrorString(_e), __FILE__, __LINE__); \
    return; } } while (0)
#define NCCL_CHECK(expr) do { ncclResult_t _r = (expr); if (_r != ncclSuccess) { \
    fprintf(stderr, "RCCL error %s at %s:%d\n", ncclGetErrorString(_r), __FILE__, __LINE__); \
    return; } } while (0)

namespace ag {
#define LAS __attribute__((address_space(3)))
typedef unsigned short bf16_t;
typedef short bf16x8 __attribute__((ext_vector_type(8)));
typedef short bf16x4 __attribute__((ext_vector_type(4)));
typedef float f32x4 __attribute__((ext_vector_type(4)));
typedef unsigned u32x4 __attribute__((ext_vector_type(4)));
typedef unsigned u32x2 __attribute__((ext_vector_type(2)));

constexpr int GM = 4096, GK = 4096, GN = 4096;
constexpr int BM = 256, BK = 64, HALF = 128, HTB = HALF * BK * 2, STAGE_BYTES = 8 * HTB, NXCD = 8, WGM = 8;
constexpr int NT = GK / BK;

__device__ __forceinline__ int lds_byte(int r, int c) { const int st = (r >> 4) * 2 + (c >> 5), rr = r & 15, cc = c & 31, ob = rr * 64 + cc * 2; return st * 1024 + (ob ^ (((ob >> 9) & 1) << 5)); }
__device__ __forceinline__ void stage_rc(int b, int& R, int& C) { const int st = b / 1024, sb = b % 1024, swz = sb ^ (((sb >> 9) & 1) << 5); R = (st >> 1) * 16 + swz / 64; C = (st & 1) * 32 + (swz % 64) / 2; }
__device__ __forceinline__ unsigned cvt_pk_bf16(float lo, float hi) { unsigned r; asm volatile("v_cvt_pk_bf16_f32 %0, %1, %2" : "=v"(r) : "v"(lo), "v"(hi)); return r; }

__global__ __launch_bounds__(512, 2) void ag_gemm(const bf16_t* __restrict__ Ag, const bf16_t* __restrict__ W, bf16_t* __restrict__ O) {
    extern __shared__ __attribute__((aligned(16))) unsigned char lds_raw[];
    LAS unsigned char* lds = (LAS unsigned char*)lds_raw;
    const int tid = threadIdx.x, wid = __builtin_amdgcn_readfirstlane(tid >> 6), lane = tid & 63, wr = wid >> 2, wc = wid & 3, fr = lane & 15, fq = lane >> 4;
    int pm, pn;
    { const int nN = GN / BM; int wgid = blockIdx.x; const int nwg = (GM / BM) * nN; const int q = nwg / NXCD, xcd = wgid % NXCD, off = wgid / NXCD; wgid = xcd * q + off;
      const int nig = WGM * nN; pm = (wgid / nig) * WGM + (wgid % nig) % WGM; pn = (wgid % nig) / WGM; }
    unsigned voffA[2], voffB[2];
#pragma unroll
    for (int i = 0; i < 2; ++i) { int R, C; stage_rc(tid * 16 + i * 8192, R, C); voffA[i] = (unsigned)(R * GK + C) * 2u;
        const int b = tid * 16 + i * 8192, row = b >> 8, segp = (b & 255) >> 5, f = (row & 3) | (((row >> 3) & 1) << 2), col = ((segp ^ f) << 4) + ((b >> 4) & 1) * 8;
        voffB[i] = (unsigned)(row * GN + col) * 2u; }
    const size_t kstepA = (size_t)(BK * 2), hstepA = (size_t)HALF * GK * 2;
    const size_t kstepB = (size_t)BK * GN * 2, hstepB = (size_t)HALF * 2;
    const unsigned ldsw = (unsigned)wid * 1024u;
    const int aoff = lds_byte(wr * 64 + fr, fq * 8);
    int boff[2];
    { const int g = fq, q = (lane & 15) >> 2, p = lane & 3, f = q | ((g & 1) << 2);
#pragma unroll
      for (int n = 0; n < 2; ++n) boff[n] = 256 * (8 * g + q) + 32 * ((wc * 2 + n) ^ f) + 8 * p; }
#define SA(b, h) (((b) * 2 + (h)) * HTB)
#define SB(b, h) ((4 + (b) * 2 + (h)) * HTB)
#define STAGE(bufoff, gbase, voff) do { _Pragma("unroll") for (int _i = 0; _i < 2; ++_i) \
        __builtin_amdgcn_global_load_lds((const unsigned*)((const char*)(gbase) + (voff)[_i]), (LAS unsigned*)(lds + (bufoff) + ldsw + _i * 8192), 16, 0, 0); } while (0)
#define LDA(dst, b, h) do { _Pragma("unroll") for (int m = 0; m < 4; ++m) _Pragma("unroll") for (int k = 0; k < 2; ++k) dst[m][k] = *(const LAS bf16x8*)(lds + SA(b, h) + aoff + m * 2048 + k * 1024); } while (0)
#define LDB(dst, b, h) do { _Pragma("unroll") for (int n = 0; n < 2; ++n) _Pragma("unroll") for (int k = 0; k < 2; ++k) { \
        const bf16x4 lo = __builtin_amdgcn_ds_read_tr16_b64_v4i16((LAS bf16x4*)(lds + SB(b, h) + boff[n] + k * 8192)); \
        const bf16x4 hi = __builtin_amdgcn_ds_read_tr16_b64_v4i16((LAS bf16x4*)(lds + SB(b, h) + boff[n] + k * 8192 + 1024)); \
        dst[n][k] = __builtin_shufflevector(lo, hi, 0, 1, 2, 3, 4, 5, 6, 7); } } while (0)
#define MMA(ai, bj, At, Bt) do { __builtin_amdgcn_s_setprio(1); _Pragma("unroll") for (int m = 0; m < 4; ++m) _Pragma("unroll") for (int n = 0; n < 2; ++n) _Pragma("unroll") for (int k = 0; k < 2; ++k) \
        acc[ai][bj][m][n] = __builtin_amdgcn_mfma_f32_16x16x32_bf16(Bt[n][k], At[m][k], acc[ai][bj][m][n], 0, 0, 0); __builtin_amdgcn_s_setprio(0); } while (0)
#define WAIT_V(n) asm volatile("s_waitcnt vmcnt(" #n ")" ::: "memory")
#define WAIT_L(n) asm volatile("s_waitcnt lgkmcnt(" #n ")" ::: "memory")
#define BAR __builtin_amdgcn_s_barrier()
#define SCHED __builtin_amdgcn_sched_barrier(0)
    f32x4 acc[2][2][4][2];
#pragma unroll
    for (int a = 0; a < 2; ++a)
#pragma unroll
        for (int b = 0; b < 2; ++b)
#pragma unroll
            for (int m = 0; m < 4; ++m)
#pragma unroll
                for (int n = 0; n < 2; ++n) acc[a][b][m][n] = (f32x4){0.f, 0.f, 0.f, 0.f};
    bf16x8 At[4][2], B0[2][2], B1[2][2];
    const char* cA = (const char*)Ag + (size_t)pm * BM * GK * 2; const char* cB = (const char*)W + (size_t)pn * BM * 2;
    STAGE(SB(0, 0), cB, voffB); STAGE(SA(0, 0), cA, voffA); STAGE(SB(0, 1), cB + hstepB, voffB); STAGE(SA(0, 1), cA + hstepA, voffA);
    if (wr == 1) BAR;
    WAIT_V(4); BAR;
    STAGE(SB(1, 0), cB + kstepB, voffB); STAGE(SA(1, 0), cA + kstepA, voffA); STAGE(SB(1, 1), cB + hstepB + kstepB, voffB);
    WAIT_V(6); BAR;
    for (int t = 0; t < NT; t += 2) {
        const bool last = (t == NT - 2);
        const int t2 = last ? 0 : t + 2;
        const char* a1 = cA + (size_t)(t + 1) * kstepA;
        const char* a2 = cA + (size_t)t2 * kstepA; const char* b2 = cB + (size_t)t2 * kstepB;
        const char* a3 = a2 + kstepA; const char* b3 = b2 + kstepB;
        LDB(B0, 0, 0); SCHED; LDA(At, 0, 0); STAGE(SA(1, 1), a1 + hstepA, voffA);
        WAIT_L(8); BAR; WAIT_L(0); MMA(0, 0, At, B0); BAR; SCHED;
        LDB(B1, 0, 1); STAGE(SB(0, 0), b2, voffB);
        BAR; WAIT_L(0); MMA(0, 1, At, B1); BAR;
        LDA(At, 0, 1); STAGE(SA(0, 0), a2, voffA);
        BAR; WAIT_L(0); MMA(1, 0, At, B0); BAR; SCHED;
        STAGE(SB(0, 1), b2 + hstepB, voffB);
        WAIT_V(6); BAR; MMA(1, 1, At, B1); BAR;
        LDB(B0, 1, 0); SCHED; LDA(At, 1, 0); STAGE(SA(0, 1), a2 + hstepA, voffA);
        WAIT_L(8); BAR; WAIT_L(0); MMA(0, 0, At, B0); BAR; SCHED;
        LDB(B1, 1, 1); STAGE(SB(1, 0), b3, voffB);
        BAR; WAIT_L(0); MMA(0, 1, At, B1); BAR;
        LDA(At, 1, 1); STAGE(SA(1, 0), a3, voffA);
        BAR; WAIT_L(0); MMA(1, 0, At, B0); BAR; SCHED;
        STAGE(SB(1, 1), b3 + hstepB, voffB);
        WAIT_V(6); BAR; MMA(1, 1, At, B1); BAR;
    }
    WAIT_V(0);
    if (wr == 0) BAR;
    const int row0 = pm * BM + wr * 64 + fr, col0 = pn * BM + wc * 32 + 4 * fq;
#pragma unroll
    for (int ai = 0; ai < 2; ++ai)
#pragma unroll
        for (int m = 0; m < 4; ++m) { bf16_t* rowp = O + (size_t)(row0 + ai * HALF + m * 16) * GN + col0;
#pragma unroll
            for (int bj = 0; bj < 2; ++bj)
#pragma unroll
                for (int n = 0; n < 2; ++n) { f32x4 v = acc[ai][bj][m][n];
                    v[0] = fmaxf(v[0], 0.f); v[1] = fmaxf(v[1], 0.f); v[2] = fmaxf(v[2], 0.f); v[3] = fmaxf(v[3], 0.f);
                    u32x2 w; w.x = cvt_pk_bf16(v[0], v[1]); w.y = cvt_pk_bf16(v[2], v[3]);
                    *(u32x2*)(rowp + bj * HALF + n * 16) = w; } }
}
}

extern "C" void kernel_launch(int rank, int nranks, ncclComm_t comm,
                              hipStream_t stream,
                              void* const* d_in, const int* in_sizes, int n_in,
                              void* d_out, int out_size,
                              const int* in_dtypes, int out_dtype,
                              void* const* d_out_all,
                              void* const* d_ws_all, size_t ws_bytes) {
    static bool attr_done = false;
    if (!attr_done) {
        HIP_CHECK(hipFuncSetAttribute((const void*)ag::ag_gemm, hipFuncAttributeMaxDynamicSharedMemorySize, ag::STAGE_BYTES));
        attr_done = true;
    }
    ag::bf16_t* xg = (ag::bf16_t*)d_ws_all[rank];
    NCCL_CHECK(ncclAllGather(d_in[0], xg, (size_t)512 * 4096, ncclBfloat16, comm, stream));
    ag::ag_gemm<<<256, 512, ag::STAGE_BYTES, stream>>>(xg, (const ag::bf16_t*)d_in[1], (ag::bf16_t*)d_out);
}
```

```cpp
#include <hip/hip_runtime.h>
#include <rccl/rccl.h>
#include <cstdio>

#define HIP_CHECK(expr) do { hipError_t _e = (expr); if (_e != hipSuccess) { \
    fprintf(stderr, "HIP error %s at %s:%d\n", hipGetErrorString(_e), __FILE__, __LINE__); \
    return; } } while (0)

namespace ag {
#define LAS __attribute__((address_space(3)))
typedef unsigned short bf16_t;
typedef short bf16x8 __attribute__((ext_vector_type(8)));
typedef short bf16x4 __attribute__((ext_vector_type(4)));
typedef float f32x4 __attribute__((ext_vector_type(4)));
typedef unsigned u32x4 __attribute__((ext_vector_type(4)));
typedef unsigned u32x2 __attribute__((ext_vector_type(2)));

constexpr int NRANK = 8, SH_ROWS = 512;
constexpr int GM = 4096, GK = 4096, GN = 4096;
constexpr int BM = 256, BK = 64, HALF = 128, HTB = HALF * BK * 2, STAGE_BYTES = 8 * HTB, NXCD = 8, WGM = 8;
constexpr int NT = GK / BK, NWG = (GM / BM) * (GN / BM);
constexpr int KC = 256, NCH = GK / KC, DLEAD = 2;
constexpr size_t WS_CTR = 0, WS_TMO = 128, WS_FLAGS = 4096, WS_X = 1u << 20, XBYTES = (size_t)GM * GK * 2;
constexpr unsigned long long DEADLINE_TICKS = 20000000ull;
constexpr int LDS_BYTES = STAGE_BYTES + 64;

struct Params { const bf16_t* x; const bf16_t* w; bf16_t* out; unsigned char* ws[NRANK]; int rank; };

__device__ __forceinline__ int lds_byte(int r, int c) { const int st = (r >> 4) * 2 + (c >> 5), rr = r & 15, cc = c & 31, ob = rr * 64 + cc * 2; return st * 1024 + (ob ^ (((ob >> 9) & 1) << 5)); }
__device__ __forceinline__ void stage_rc(int b, int& R, int& C) { const int st = b / 1024, sb = b % 1024, swz = sb ^ (((sb >> 9) & 1) << 5); R = (st >> 1) * 16 + swz / 64; C = (st & 1) * 32 + (swz % 64) / 2; }
__device__ __forceinline__ unsigned cvt_pk_bf16(float lo, float hi) { unsigned r; asm volatile("v_cvt_pk_bf16_f32 %0, %1, %2" : "=v"(r) : "v"(lo), "v"(hi)); return r; }

__global__ __launch_bounds__(512, 2) void ag_gemm(const Params P) {
    extern __shared__ __attribute__((aligned(16))) unsigned char lds_raw[];
    LAS unsigned char* lds = (LAS unsigned char*)lds_raw;
    const int tid = threadIdx.x, wid = __builtin_amdgcn_readfirstlane(tid >> 6), lane = tid & 63, wr = wid >> 2, wc = wid & 3, fr = lane & 15, fq = lane >> 4;
    const int rank = P.rank;
    unsigned char* const wsl = P.ws[rank];
    unsigned* const tmo = (unsigned*)(wsl + WS_TMO);
    { LAS unsigned* bc = (LAS unsigned*)(lds + STAGE_BYTES);
      if (tid == 0) { const unsigned t = __hip_atomic_fetch_add((unsigned*)(wsl + WS_CTR), 1u, __ATOMIC_RELAXED, __HIP_MEMORY_SCOPE_AGENT); bc[0] = t / (unsigned)NWG + 1u; }
      __syncthreads(); }
    const unsigned call = __builtin_amdgcn_readfirstlane(*(LAS unsigned*)(lds + STAGE_BYTES));
    const size_t xoff = WS_X + (size_t)(call & 1u) * XBYTES;
    int pm, pn;
    { const int nN = GN / BM; int wgid = blockIdx.x; const int q = NWG / NXCD, xcd = wgid % NXCD, off = wgid / NXCD; wgid = xcd * q + off;
      const int nig = WGM * nN; pm = (wgid / nig) * WGM + (wgid % nig) % WGM; pn = (wgid % nig) / WGM; }
    const int src = pm >> 1; const bool local = (src == rank);
    unsigned voffA, voffB;
    { int R, C; stage_rc((tid & 255) * 16, R, C); voffA = (unsigned)(R * GK + C) * 2u;
      const int b = (tid & 255) * 16, row = b >> 8, segp = (b & 255) >> 5, f = (row & 3) | (((row >> 3) & 1) << 2), col = ((segp ^ f) << 4) + ((b >> 4) & 1) * 8;
      voffB = (unsigned)(row * GN + col) * 2u; }
    const size_t kstepA = (size_t)(BK * 2), hstepA = (size_t)HALF * GK * 2, pstepA = (size_t)32 * GK * 2;
    const size_t kstepB = (size_t)BK * GN * 2, hstepB = (size_t)HALF * 2, pstepB = (size_t)16 * GN * 2;
    const unsigned ldsw = (unsigned)(wid & 3) * 1024u;
    const int aoff = lds_byte(wr * 64 + fr, fq * 8);
    unsigned baddr[2];
    { const int g = fq, q = (lane & 15) >> 2, p = lane & 3, f = q | ((g & 1) << 2);
#pragma unroll
      for (int n = 0; n < 2; ++n) baddr[n] = (unsigned)(size_t)lds + 4 * HTB + 256 * (8 * g + q) + 32 * ((wc * 2 + n) ^ f) + 8 * p; }
#define SA(b, h) (((b) * 2 + (h)) * HTB)
#define SB(b, h) ((4 + (b) * 2 + (h)) * HTB)
#define STAGE(bufoff, gbase, voff, pstep) do { if (wr == 0) { _Pragma("unroll") for (int _i = 0; _i < 4; ++_i) \
        __builtin_amdgcn_global_load_lds((const unsigned*)((const char*)(gbase) + _i * (pstep) + (voff)), (LAS unsigned*)(lds + (bufoff) + ldsw + _i * 4096), 16, 0, 0); } } while (0)
#define STA(bufoff, gbase) STAGE(bufoff, gbase, voffA, pstepA)
#define STB(bufoff, gbase) STAGE(bufoff, gbase, voffB, pstepB)
#define LDA(dst, b, h) do { _Pragma("unroll") for (int m = 0; m < 4; ++m) _Pragma("unroll") for (int k = 0; k < 2; ++k) dst[m][k] = *(const LAS bf16x8*)(lds + SA(b, h) + aoff + m * 2048 + k * 1024); } while (0)
#define TRRD(d, n, off) asm volatile("ds_read_b64_tr_b16 %0, %1 offset:%2" : "=v"(d) : "v"(baddr[n]), "i"(off))
#define LDB(dst, b, h) do { _Pragma("unroll") for (int n = 0; n < 2; ++n) _Pragma("unroll") for (int k = 0; k < 2; ++k) { \
        bf16x4 lo, hi; TRRD(lo, n, ((b) * 2 + (h)) * HTB + k * 8192); TRRD(hi, n, ((b) * 2 + (h)) * HTB + k * 8192 + 1024); \
        dst[n][k] = __builtin_shufflevector(lo, hi, 0, 1, 2, 3, 4, 5, 6, 7); } } while (0)
#define WAIT_LB(Bx) asm volatile("s_waitcnt lgkmcnt(0)" : "+v"(Bx[0][0]), "+v"(Bx[0][1]), "+v"(Bx[1][0]), "+v"(Bx[1][1]) :: "memory")
#define MMA(ai, bj, At, Bt) do { __builtin_amdgcn_s_setprio(1); _Pragma("unroll") for (int m = 0; m < 4; ++m) _Pragma("unroll") for (int n = 0; n < 2; ++n) _Pragma("unroll") for (int k = 0; k < 2; ++k) \
        acc[ai][bj][m][n] = __builtin_amdgcn_mfma_f32_16x16x32_bf16(Bt[n][k], At[m][k], acc[ai][bj][m][n], 0, 0, 0); __builtin_amdgcn_s_setprio(0); } while (0)
#define WAIT_V(n) asm volatile("s_waitcnt vmcnt(" #n ")" ::: "memory")
#define WAIT_L(n) asm volatile("s_waitcnt lgkmcnt(" #n ")" ::: "memory")
#define BAR __builtin_amdgcn_s_barrier()
#define SCHED __builtin_amdgcn_sched_barrier(0)

    unsigned wsv_lo, wsv_hi;
    { unsigned long long v = (unsigned long long)P.ws[0];
#pragma unroll
      for (int p = 1; p < NRANK; ++p) v = (lane & 7) == p ? (unsigned long long)P.ws[p] : v;
      wsv_lo = (unsigned)v; wsv_hi = (unsigned)(v >> 32); }
    const unsigned pvoff = (unsigned)((rank * SH_ROWS + 2 * blockIdx.x + (lane >> 5)) * GK + (lane & 31) * 8) * 2u;
    auto peer_k = [&](int k) -> int { return (rank + 1 + (k + (int)blockIdx.x) % (NRANK - 1)) & (NRANK - 1); };
    auto peer_ws = [&](int p) -> unsigned char* { return (unsigned char*)(((unsigned long long)(unsigned)__builtin_amdgcn_readlane((int)wsv_hi, p) << 32) | (unsigned)__builtin_amdgcn_readlane((int)wsv_lo, p)); };
    u32x4 piece = (u32x4){0u, 0u, 0u, 0u};
    auto load_piece = [&](u32x4& dst, int c) { const char* base = (const char*)P.x - (size_t)rank * SH_ROWS * GK * 2 + (size_t)c * KC * 2;
        asm volatile("global_load_dwordx4 %0, %1, %2" : "=v"(dst) : "v"(pvoff), "s"(base) : "memory"); };
    auto push_one = [&](int c, int k, u32x4 v) {
        const unsigned char* base = peer_ws(peer_k(k)) + xoff + (size_t)c * KC * 2;
        asm volatile("global_store_dwordx4 %0, %1, %2 sc0 sc1\n\ts_nop 1" :: "v"(pvoff), "v"(v), "s"(base) : "memory");
        };
    auto publish = [&](int c) {
        asm volatile("s_waitcnt vmcnt(0)" ::: "memory");
        if (lane < NRANK && lane != rank) { const unsigned* fp = (const unsigned*)((((unsigned long long)wsv_hi << 32) | wsv_lo) + WS_FLAGS) + ((rank * NCH + c) * NWG + blockIdx.x);
            asm volatile("global_store_dword %0, %1, off sc0 sc1\n\ts_nop 1" :: "v"(fp), "v"(call) : "memory"); } };
    auto flag_ptr = [&](int c) -> const unsigned long long* { return (const unsigned long long*)((const unsigned*)(wsl + WS_FLAGS) + ((src * NCH + c) * NWG + (pm & 1) * 128)) + lane; };
    auto flags_ok = [&](unsigned long long v) -> bool { return __all((unsigned)v >= call && (unsigned)(v >> 32) >= call); };
    auto sys_load64 = [&](const unsigned long long* p) -> unsigned long long { unsigned long long v; asm volatile("global_load_dwordx2 %0, %1, off sc0 sc1\n\ts_waitcnt vmcnt(0)" : "=&v"(v) : "v"(p) : "memory"); return v; };
    auto tmo_load = [&]() -> unsigned { unsigned v; asm volatile("global_load_dword %0, %1, off sc1\n\ts_waitcnt vmcnt(0)" : "=&v"(v) : "v"(tmo) : "memory"); return __builtin_amdgcn_readfirstlane(v); };
    auto wait_chunk = [&](int c) {
        const unsigned long long* fl = flag_ptr(c);
        const unsigned long long t0 = __builtin_amdgcn_s_memrealtime();
        for (;;) {
            if (flags_ok(sys_load64(fl))) break;
            if (tmo_load() == call || __builtin_amdgcn_s_memrealtime() - t0 > DEADLINE_TICKS) {
                asm volatile("global_store_dword %0, %1, off sc1\n\ts_nop 1" :: "v"(tmo), "v"(call) : "memory");
                break; }
            __builtin_amdgcn_s_sleep(1);
        } };
    auto acquire = [&]() {
        asm volatile("buffer_inv sc0 sc1\n\ts_waitcnt vmcnt(0)" ::: "memory");
        };
    unsigned long long early = 0;
    auto poll_issue = [&](int c) { const unsigned long long* fl = flag_ptr(c); asm volatile("global_load_dwordx2 %0, %1, off sc0 sc1" : "=v"(early) : "v"(fl) : "memory"); };
    auto poll_check = [&](int c) { asm volatile("s_waitcnt vmcnt(0)" : "+v"(early) :: "memory");
        if (!flags_ok(early)) wait_chunk(c);
        acquire(); };

    f32x4 acc[2][2][4][2];
#pragma unroll
    for (int a = 0; a < 2; ++a)
#pragma unroll
        for (int b = 0; b < 2; ++b)
#pragma unroll
            for (int m = 0; m < 4; ++m)
#pragma unroll
                for (int n = 0; n < 2; ++n) acc[a][b][m][n] = (f32x4){0.f, 0.f, 0.f, 0.f};
    bf16x8 At[4][2], B0[2][2], B1[2][2];
    const char* cA = local ? (const char*)P.x + (size_t)(pm & 1) * BM * GK * 2 : (const char*)(wsl + xoff) + (size_t)pm * BM * GK * 2;
    const char* cB = (const char*)P.w + (size_t)pn * BM * 2;
#define TP(acc, code) do { code; } while (0)
    STB(SB(0, 0), cB); STB(SB(0, 1), cB + hstepB);
    if (wid == 4) {
        u32x4 p0, p1; load_piece(p0, 0); load_piece(p1, 1); load_piece(piece, 2);
        asm volatile("s_waitcnt vmcnt(0)" : "+v"(p0), "+v"(p1), "+v"(piece) :: "memory");
        for (int k = 0; k < NRANK - 1; ++k) push_one(0, k, p0);
        publish(0);
        for (int k = 0; k < NRANK - 1; ++k) push_one(1, k, p1);
        publish(1);
        if (!local) { wait_chunk(0); acquire(); }
    }
    asm volatile("" ::: "memory"); BAR; asm volatile("" ::: "memory");
    STA(SA(0, 0), cA); STA(SA(0, 1), cA + hstepA);
    if (wr == 1) BAR;
    WAIT_V(4); BAR;
    STB(SB(1, 0), cB + kstepB); STA(SA(1, 0), cA + kstepA); STB(SB(1, 1), cB + hstepB + kstepB);
    WAIT_V(12); BAR;
    for (int t = 0; t < NT; t += 2) {
        const bool last = (t == NT - 2);
        const int t2 = last ? 0 : t + 2;
        const char* a1 = cA + (size_t)(t + 1) * kstepA;
        const char* a2 = cA + (size_t)t2 * kstepA; const char* b2 = cB + (size_t)t2 * kstepB;
        const char* a3 = a2 + kstepA; const char* b3 = b2 + kstepB;
        const int it = t >> 1, cp = (it >> 1) + DLEAD, cn = (it >> 1) + 1;
        const bool cw = (wid == 4), ev = (it & 1) == 0, dopush = cw && ev && cp < NCH;
#define PUSH(k) do { if (dopush) TP(tp_push, push_one(cp, k, piece)); } while (0)
        if (cw && !ev && cn < NCH && !local) TP(tp_wait, poll_check(cn));
        if (cw && ev) asm volatile("s_waitcnt vmcnt(0)" : "+v"(piece) :: "memory");
        PUSH(0);
        LDB(B0, 0, 0); SCHED; LDA(At, 0, 0); STA(SA(1, 1), a1 + hstepA);
        WAIT_L(8); BAR; WAIT_LB(B0); MMA(0, 0, At, B0); BAR; SCHED;
        PUSH(1);
        LDB(B1, 0, 1); STB(SB(0, 0), b2);
        BAR; WAIT_LB(B1); MMA(0, 1, At, B1); BAR;
        PUSH(2);
        LDA(At, 0, 1); STA(SA(0, 0), a2);
        BAR; WAIT_L(0); MMA(1, 0, At, B0); BAR; SCHED;
        PUSH(3);
        STB(SB(0, 1), b2 + hstepB);
        WAIT_V(12); BAR; MMA(1, 1, At, B1); BAR;
        PUSH(4);
        if (cw && !ev && cp < NCH) TP(tp_pub, publish(cp));
        LDB(B0, 1, 0); SCHED; LDA(At, 1, 0); STA(SA(0, 1), a2 + hstepA);
        WAIT_L(8); BAR; WAIT_LB(B0); MMA(0, 0, At, B0); BAR; SCHED;
        PUSH(5);
        LDB(B1, 1, 1); STB(SB(1, 0), b3);
        BAR; WAIT_LB(B1); MMA(0, 1, At, B1); BAR;
        PUSH(6);
        LDA(At, 1, 1); STA(SA(1, 0), a3);
        BAR; WAIT_L(0); MMA(1, 0, At, B0); BAR; SCHED;
        if (cw && !ev && cp + 1 < NCH) load_piece(piece, cp + 1);
        if (cw && ev && cn < NCH && !local) poll_issue(cn);
        STB(SB(1, 1), b3 + hstepB);
        WAIT_V(12); BAR; MMA(1, 1, At, B1); BAR;
    }
    WAIT_V(0);
    if (wr == 0) BAR;
    const bool dead = tmo_load() == call;
    const int row0 = pm * BM + wr * 64 + fr, col0 = pn * BM + wc * 32 + 4 * fq;
#pragma unroll
    for (int ai = 0; ai < 2; ++ai)
#pragma unroll
        for (int m = 0; m < 4; ++m) { bf16_t* rowp = P.out + (size_t)(row0 + ai * HALF + m * 16) * GN + col0;
#pragma unroll
            for (int bj = 0; bj < 2; ++bj)
#pragma unroll
                for (int n = 0; n < 2; ++n) { f32x4 v = acc[ai][bj][m][n];
                    v[0] = fmaxf(v[0], 0.f); v[1] = fmaxf(v[1], 0.f); v[2] = fmaxf(v[2], 0.f); v[3] = fmaxf(v[3], 0.f);
                    u32x2 w; w.x = cvt_pk_bf16(v[0], v[1]); w.y = cvt_pk_bf16(v[2], v[3]);
                    if (dead) { w.x = 0x7fc07fc0u; w.y = 0x7fc07fc0u; }
                    *(u32x2*)(rowp + bj * HALF + n * 16) = w; } }
}
}

extern "C" void kernel_launch(int rank, int nranks, ncclComm_t comm,
                              hipStream_t stream,
                              void* const* d_in, const int* in_sizes, int n_in,
                              void* d_out, int out_size,
                              const int* in_dtypes, int out_dtype,
                              void* const* d_out_all,
                              void* const* d_ws_all, size_t ws_bytes) {
    if (nranks != ag::NRANK || (unsigned)rank >= (unsigned)nranks || ws_bytes < ag::WS_X + 2 * ag::XBYTES) { fprintf(stderr, "unexpected problem shape\n"); return; }
    HIP_CHECK(hipFuncSetAttribute((const void*)ag::ag_gemm, hipFuncAttributeMaxDynamicSharedMemorySize, ag::LDS_BYTES));
    ag::Params P;
    P.x = (const ag::bf16_t*)d_in[0]; P.w = (const ag::bf16_t*)d_in[1]; P.out = (ag::bf16_t*)d_out; P.rank = rank;
    for (int p = 0; p < ag::NRANK; ++p) P.ws[p] = (unsigned char*)d_ws_all[p];
    ag::ag_gemm<<<ag::NWG, 512, ag::LDS_BYTES, stream>>>(P);
}
```

```cpp
#include <hip/hip_runtime.h>
#include <rccl/rccl.h>
#include <cstdio>

#define HIP_CHECK(expr) do { hipError_t _e = (expr); if (_e != hipSuccess) { \
    fprintf(stderr, "HIP error %s at %s:%d\n", hipGetErrorString(_e), __FILE__, __LINE__); \
    return; } } while (0)

namespace ag {
#define LAS __attribute__((address_space(3)))
typedef unsigned short bf16_t;
typedef short bf16x8 __attribute__((ext_vector_type(8)));
typedef short bf16x4 __attribute__((ext_vector_type(4)));
typedef float f32x4 __attribute__((ext_vector_type(4)));
typedef unsigned u32x4 __attribute__((ext_vector_type(4)));
typedef unsigned u32x2 __attribute__((ext_vector_type(2)));

constexpr int NRANK = 8, SH_ROWS = 512;
constexpr int GM = 4096, GK = 4096, GN = 4096;
constexpr int BM = 256, BK = 64, HALF = 128, HTB = HALF * BK * 2, STAGE_BYTES = 8 * HTB, NXCD = 8, WGM = 8;
constexpr int NT = GK / BK, NWG = (GM / BM) * (GN / BM);
constexpr int KC = 256, NCH = GK / KC, DLEAD = 2;
constexpr size_t WS_CTR = 0, WS_TMO = 128, WS_FLAGS = 4096, WS_X = 1u << 20, XBYTES = (size_t)GM * GK * 2;
constexpr unsigned long long DEADLINE_TICKS = 20000000ull;
constexpr int LDS_BYTES = STAGE_BYTES + 64;

struct Params { const bf16_t* x; const bf16_t* w; bf16_t* out; unsigned char* ws[NRANK]; int rank; };

__device__ __forceinline__ int lds_byte(int r, int c) { const int st = (r >> 4) * 2 + (c >> 5), rr = r & 15, cc = c & 31, ob = rr * 64 + cc * 2; return st * 1024 + (ob ^ (((ob >> 9) & 1) << 5)); }
__device__ __forceinline__ void stage_rc(int b, int& R, int& C) { const int st = b / 1024, sb = b % 1024, swz = sb ^ (((sb >> 9) & 1) << 5); R = (st >> 1) * 16 + swz / 64; C = (st & 1) * 32 + (swz % 64) / 2; }
__device__ __forceinline__ unsigned cvt_pk_bf16(float lo, float hi) { unsigned r; asm volatile("v_cvt_pk_bf16_f32 %0, %1, %2" : "=v"(r) : "v"(lo), "v"(hi)); return r; }

__global__ __launch_bounds__(512, 2) void ag_gemm(const Params P) {
    extern __shared__ __attribute__((aligned(16))) unsigned char lds_raw[];
    LAS unsigned char* lds = (LAS unsigned char*)lds_raw;
    const int tid = threadIdx.x, wid = __builtin_amdgcn_readfirstlane(tid >> 6), lane = tid & 63, wr = wid >> 2, wc = wid & 3, fr = lane & 15, fq = lane >> 4;
    const int rank = P.rank;
    unsigned char* const wsl = P.ws[rank];
    unsigned* const tmo = (unsigned*)(wsl + WS_TMO);
    { LAS unsigned* bc = (LAS unsigned*)(lds + STAGE_BYTES);
      if (tid == 0) { const unsigned t = __hip_atomic_fetch_add((unsigned*)(wsl + WS_CTR), 1u, __ATOMIC_RELAXED, __HIP_MEMORY_SCOPE_AGENT); bc[0] = t / (unsigned)NWG + 1u; }
      __syncthreads(); }
    const unsigned call = __builtin_amdgcn_readfirstlane(*(LAS unsigned*)(lds + STAGE_BYTES));
    const size_t xoff = WS_X + (size_t)(call & 1u) * XBYTES;
    int pm, pn;
    { const int nN = GN / BM; int wgid = blockIdx.x; const int q = NWG / NXCD, xcd = wgid % NXCD, off = wgid / NXCD; wgid = xcd * q + off;
      const int nig = WGM * nN; pm = (wgid / nig) * WGM + (wgid % nig) % WGM; pn = (wgid % nig) / WGM; }
    const int src = pm >> 1; const bool local = (src == rank);
    unsigned voffA, voffB;
    { int R, C; stage_rc((tid & 255) * 16, R, C); voffA = (unsigned)(R * GK + C) * 2u;
      const int b = (tid & 255) * 16, row = b >> 8, segp = (b & 255) >> 5, f = (row & 3) | (((row >> 3) & 1) << 2), col = ((segp ^ f) << 4) + ((b >> 4) & 1) * 8;
      voffB = (unsigned)(row * GN + col) * 2u; }
    const size_t kstepA = (size_t)(BK * 2), hstepA = (size_t)HALF * GK * 2, pstepA = (size_t)32 * GK * 2;
    const size_t kstepB = (size_t)BK * GN * 2, hstepB = (size_t)HALF * 2, pstepB = (size_t)16 * GN * 2;
    const unsigned ldsw = (unsigned)(wid & 3) * 1024u;
    const int aoff = lds_byte(wr * 64 + fr, fq * 8);
    unsigned baddr[2];
    { const int g = fq, q = (lane & 15) >> 2, p = lane & 3, f = q | ((g & 1) << 2);
#pragma unroll
      for (int n = 0; n < 2; ++n) baddr[n] = (unsigned)(size_t)lds + 4 * HTB + 256 * (8 * g + q) + 32 * ((wc * 2 + n) ^ f) + 8 * p; }
#define SA(b, h) (((b) * 2 + (h)) * HTB)
#define SB(b, h) ((4 + (b) * 2 + (h)) * HTB)
#define STAGE(bufoff, gbase, voff, pstep) do { if (wr == 0) { _Pragma("unroll") for (int _i = 0; _i < 4; ++_i) \
        __builtin_amdgcn_global_load_lds((const unsigned*)((const char*)(gbase) + _i * (pstep) + (voff)), (LAS unsigned*)(lds + (bufoff) + ldsw + _i * 4096), 16, 0, 0); } } while (0)
#define STA(bufoff, gbase) STAGE(bufoff, gbase, voffA, pstepA)
#define STB(bufoff, gbase) STAGE(bufoff, gbase, voffB, pstepB)
#define LDA(dst, b, h) do { _Pragma("unroll") for (int m = 0; m < 4; ++m) _Pragma("unroll") for (int k = 0; k < 2; ++k) dst[m][k] = *(const LAS bf16x8*)(lds + SA(b, h) + aoff + m * 2048 + k * 1024); } while (0)
#define TRRD(d, n, off) asm volatile("ds_read_b64_tr_b16 %0, %1 offset:%2" : "=v"(d) : "v"(baddr[n]), "i"(off))
#define LDB(dst, b, h) do { _Pragma("unroll") for (int n = 0; n < 2; ++n) _Pragma("unroll") for (int k = 0; k < 2; ++k) { \
        bf16x4 lo, hi; TRRD(lo, n, ((b) * 2 + (h)) * HTB + k * 8192); TRRD(hi, n, ((b) * 2 + (h)) * HTB + k * 8192 + 1024); \
        dst[n][k] = __builtin_shufflevector(lo, hi, 0, 1, 2, 3, 4, 5, 6, 7); } } while (0)
#define WAIT_LB(Bx) asm volatile("s_waitcnt lgkmcnt(0)" : "+v"(Bx[0][0]), "+v"(Bx[0][1]), "+v"(Bx[1][0]), "+v"(Bx[1][1]) :: "memory")
#define MMA(ai, bj, At, Bt) do { __builtin_amdgcn_s_setprio(1); _Pragma("unroll") for (int m = 0; m < 4; ++m) _Pragma("unroll") for (int n = 0; n < 2; ++n) _Pragma("unroll") for (int k = 0; k < 2; ++k) \
        acc[ai][bj][m][n] = __builtin_amdgcn_mfma_f32_16x16x32_bf16(Bt[n][k], At[m][k], acc[ai][bj][m][n], 0, 0, 0); __builtin_amdgcn_s_setprio(0); } while (0)
#define WAIT_V(n) asm volatile("s_waitcnt vmcnt(" #n ")" ::: "memory")
#define WAIT_L(n) asm volatile("s_waitcnt lgkmcnt(" #n ")" ::: "memory")
#define BAR __builtin_amdgcn_s_barrier()
#define SCHED __builtin_amdgcn_sched_barrier(0)

    unsigned wsv_lo, wsv_hi;
    { unsigned long long v = (unsigned long long)P.ws[0];
#pragma unroll
      for (int p = 1; p < NRANK; ++p) v = (lane & 7) == p ? (unsigned long long)P.ws[p] : v;
      wsv_lo = (unsigned)v; wsv_hi = (unsigned)(v >> 32); }
    const unsigned pvoff = (unsigned)((rank * SH_ROWS + 2 * blockIdx.x + (lane >> 5)) * GK + (lane & 31) * 8) * 2u;
    auto peer_k = [&](int k) -> int { return (rank + 1 + (k + (int)blockIdx.x) % (NRANK - 1)) & (NRANK - 1); };
    auto peer_ws = [&](int p) -> unsigned char* { return (unsigned char*)(((unsigned long long)(unsigned)__builtin_amdgcn_readlane((int)wsv_hi, p) << 32) | (unsigned)__builtin_amdgcn_readlane((int)wsv_lo, p)); };
    u32x4 piece = (u32x4){0u, 0u, 0u, 0u};
    auto load_piece = [&](u32x4& dst, int c) { const char* base = (const char*)P.x - (size_t)rank * SH_ROWS * GK * 2 + (size_t)c * KC * 2;
        asm volatile("global_load_dwordx4 %0, %1, %2" : "=v"(dst) : "v"(pvoff), "s"(base) : "memory"); };
    auto push_one = [&](int c, int k, u32x4 v) {
        const unsigned char* base = peer_ws(peer_k(k)) + xoff + (size_t)c * KC * 2;
        asm volatile("global_store_dwordx4 %0, %1, %2 sc0 sc1\n\ts_nop 1" :: "v"(pvoff), "v"(v), "s"(base) : "memory");
        };
    auto publish = [&](int c) {
        asm volatile("s_waitcnt vmcnt(0)" ::: "memory");
        if (lane < NRANK && lane != rank) { const unsigned* fp = (const unsigned*)((((unsigned long long)wsv_hi << 32) | wsv_lo) + WS_FLAGS) + ((rank * NCH + c) * NWG + blockIdx.x);
            asm volatile("global_store_dword %0, %1, off sc0 sc1\n\ts_nop 1" :: "v"(fp), "v"(call) : "memory"); } };
    auto flag_ptr = [&](int c) -> const unsigned long long* { return (const unsigned long long*)((const unsigned*)(wsl + WS_FLAGS) + ((src * NCH + c) * NWG + (pm & 1) * 128)) + lane; };
    auto flags_ok = [&](unsigned long long v) -> bool { return __all((unsigned)v >= call && (unsigned)(v >> 32) >= call); };
    auto sys_load64 = [&](const unsigned long long* p) -> unsigned long long { unsigned long long v; asm volatile("global_load_dwordx2 %0, %1, off sc0 sc1\n\ts_waitcnt vmcnt(0)" : "=&v"(v) : "v"(p) : "memory"); return v; };
    auto tmo_load = [&]() -> unsigned { unsigned v; asm volatile("global_load_dword %0, %1, off sc1\n\ts_waitcnt vmcnt(0)" : "=&v"(v) : "v"(tmo) : "memory"); return __builtin_amdgcn_readfirstlane(v); };
    auto wait_chunk = [&](int c) {
        const unsigned long long* fl = flag_ptr(c);
        const unsigned long long t0 = __builtin_amdgcn_s_memrealtime();
        for (;;) {
            if (flags_ok(sys_load64(fl))) break;
            if (tmo_load() == call || __builtin_amdgcn_s_memrealtime() - t0 > DEADLINE_TICKS) {
                asm volatile("global_store_dword %0, %1, off sc1\n\ts_nop 1" :: "v"(tmo), "v"(call) : "memory");
                break; }
            __builtin_amdgcn_s_sleep(1);
        } };
    auto acquire = [&]() {
        asm volatile("buffer_inv sc0 sc1\n\ts_waitcnt vmcnt(0)" ::: "memory");
        };
    unsigned long long early = 0;
    auto poll_issue = [&](int c) { const unsigned long long* fl = flag_ptr(c); asm volatile("global_load_dwordx2 %0, %1, off sc0 sc1" : "=v"(early) : "v"(fl) : "memory"); };
    auto poll_check = [&](int c) { asm volatile("s_waitcnt vmcnt(0)" : "+v"(early) :: "memory");
        if (!flags_ok(early)) wait_chunk(c);
        };

    f32x4 acc[2][2][4][2];
#pragma unroll
    for (int a = 0; a < 2; ++a)
#pragma unroll
        for (int b = 0; b < 2; ++b)
#pragma unroll
            for (int m = 0; m < 4; ++m)
#pragma unroll
                for (int n = 0; n < 2; ++n) acc[a][b][m][n] = (f32x4){0.f, 0.f, 0.f, 0.f};
    bf16x8 At[4][2], B0[2][2], B1[2][2];
    unsigned tmo_v = 0;
    const char* cA = local ? (const char*)P.x + (size_t)(pm & 1) * BM * GK * 2 : (const char*)(wsl + xoff) + (size_t)pm * BM * GK * 2;
    const char* cB = (const char*)P.w + (size_t)pn * BM * 2;
#define TP(acc, code) do { code; } while (0)
    STB(SB(0, 0), cB); STB(SB(0, 1), cB + hstepB);
    if (wid == 4) {
        u32x4 p0, p1; load_piece(p0, 0); load_piece(p1, 1); load_piece(piece, 2);
        asm volatile("s_waitcnt vmcnt(0)" : "+v"(p0), "+v"(p1), "+v"(piece) :: "memory");
        for (int k = 0; k < NRANK - 1; ++k) push_one(0, k, p0);
        publish(0);
        for (int k = 0; k < NRANK - 1; ++k) push_one(1, k, p1);
        publish(1);
        if (!local) { wait_chunk(0); acquire(); }
    }
    asm volatile("" ::: "memory"); BAR; asm volatile("" ::: "memory");
    STA(SA(0, 0), cA); STA(SA(0, 1), cA + hstepA);
    if (wr == 1) BAR;
    WAIT_V(4); BAR;
    STB(SB(1, 0), cB + kstepB); STA(SA(1, 0), cA + kstepA); STB(SB(1, 1), cB + hstepB + kstepB);
    WAIT_V(12); BAR;
    for (int t = 0; t < NT; t += 2) {
        const bool last = (t == NT - 2);
        const int t2 = last ? 0 : t + 2;
        const char* a1 = cA + (size_t)(t + 1) * kstepA;
        const char* a2 = cA + (size_t)t2 * kstepA; const char* b2 = cB + (size_t)t2 * kstepB;
        const char* a3 = a2 + kstepA; const char* b3 = b2 + kstepB;
        const int it = t >> 1, cp = (it >> 1) + DLEAD, cn = (it >> 1) + 1;
        const bool cw = (wid == 4), ev = (it & 1) == 0, dopush = cw && ev && cp < NCH;
#define PUSH(k) do { if (dopush) TP(tp_push, push_one(cp, k, piece)); } while (0)
        if (last) asm volatile("global_load_dword %0, %1, off sc1" : "=v"(tmo_v) : "v"(tmo) : "memory");
        if (cw && !ev && cn < NCH && !local) TP(tp_wait, poll_check(cn));
        if (cw && ev) asm volatile("s_waitcnt vmcnt(0)" : "+v"(piece) :: "memory");
        PUSH(0);
        LDB(B0, 0, 0); SCHED; LDA(At, 0, 0); STA(SA(1, 1), a1 + hstepA);
        WAIT_L(8); BAR; WAIT_LB(B0); MMA(0, 0, At, B0); BAR; SCHED;
        PUSH(1);
        LDB(B1, 0, 1); STB(SB(0, 0), b2);
        BAR; WAIT_LB(B1); MMA(0, 1, At, B1); BAR;
        PUSH(2);
        LDA(At, 0, 1); STA(SA(0, 0), a2);
        BAR; WAIT_L(0); MMA(1, 0, At, B0); BAR; SCHED;
        PUSH(3);
        STB(SB(0, 1), b2 + hstepB);
        WAIT_V(12); BAR; MMA(1, 1, At, B1); BAR;
        PUSH(4);
        if (cw && !ev && cp < NCH) TP(tp_pub, publish(cp));
        LDB(B0, 1, 0); SCHED; LDA(At, 1, 0); STA(SA(0, 1), a2 + hstepA);
        WAIT_L(8); BAR; WAIT_LB(B0); MMA(0, 0, At, B0); BAR; SCHED;
        PUSH(5);
        LDB(B1, 1, 1); STB(SB(1, 0), b3);
        BAR; WAIT_LB(B1); MMA(0, 1, At, B1); BAR;
        PUSH(6);
        LDA(At, 1, 1); STA(SA(1, 0), a3);
        BAR; WAIT_L(0); MMA(1, 0, At, B0); BAR; SCHED;
        if (cw && !ev && cp + 1 < NCH) load_piece(piece, cp + 1);
        if (cw && ev && cn < NCH && !local) poll_issue(cn);
        STB(SB(1, 1), b3 + hstepB);
        WAIT_V(12); BAR; MMA(1, 1, At, B1); BAR;
    }
    asm volatile("s_waitcnt vmcnt(0)" : "+v"(tmo_v) :: "memory");
    if (wr == 0) BAR;
    const bool dead = __builtin_amdgcn_readfirstlane(tmo_v) == call;
    const int row0 = pm * BM + wr * 64 + fr, col0 = pn * BM + wc * 32 + (fq & 1) * 16 + (fq >> 1) * 8;
#pragma unroll
    for (int ai = 0; ai < 2; ++ai)
#pragma unroll
        for (int m = 0; m < 4; ++m) { bf16_t* rowp = P.out + (size_t)(row0 + ai * HALF + m * 16) * GN + col0;
#pragma unroll
            for (int bj = 0; bj < 2; ++bj) { f32x4 v0 = acc[ai][bj][m][0], v1 = acc[ai][bj][m][1];
#pragma unroll
                for (int j = 0; j < 4; ++j) { v0[j] = fmaxf(v0[j], 0.f); v1[j] = fmaxf(v1[j], 0.f); }
                const unsigned a0 = cvt_pk_bf16(v0[0], v0[1]), a1 = cvt_pk_bf16(v0[2], v0[3]), b0 = cvt_pk_bf16(v1[0], v1[1]), b1 = cvt_pk_bf16(v1[2], v1[3]);
                const auto s0 = __builtin_amdgcn_permlane16_swap(a0, b0, false, false), s1 = __builtin_amdgcn_permlane16_swap(a1, b1, false, false);
                u32x4 w; w.x = s0[0]; w.y = s1[0]; w.z = s0[1]; w.w = s1[1];
                if (dead) w = (u32x4){0x7fc07fc0u, 0x7fc07fc0u, 0x7fc07fc0u, 0x7fc07fc0u};
                *(u32x4*)(rowp + bj * HALF) = w; } }
}
}

extern "C" void kernel_launch(int rank, int nranks, ncclComm_t comm,
                              hipStream_t stream,
                              void* const* d_in, const int* in_sizes, int n_in,
                              void* d_out, int out_size,
                              const int* in_dtypes, int out_dtype,
                              void* const* d_out_all,
                              void* const* d_ws_all, size_t ws_bytes) {
    if (nranks != ag::NRANK || (unsigned)rank >= (unsigned)nranks || ws_bytes < ag::WS_X + 2 * ag::XBYTES) { fprintf(stderr, "unexpected problem shape\n"); return; }
    HIP_CHECK(hipFuncSetAttribute((const void*)ag::ag_gemm, hipFuncAttributeMaxDynamicSharedMemorySize, ag::LDS_BYTES));
    ag::Params P;
    P.x = (const ag::bf16_t*)d_in[0]; P.w = (const ag::bf16_t*)d_in[1]; P.out = (ag::bf16_t*)d_out; P.rank = rank;
    for (int p = 0; p < ag::NRANK; ++p) P.ws[p] = (unsigned char*)d_ws_all[p];
    ag::ag_gemm<<<ag::NWG, 512, ag::LDS_BYTES, stream>>>(P);
}
```

```cpp
#include <hip/hip_runtime.h>
#include <rccl/rccl.h>
#include <cstdio>

#define HIP_CHECK(expr) do { hipError_t _e = (expr); if (_e != hipSuccess) { \
    fprintf(stderr, "HIP error %s at %s:%d\n", hipGetErrorString(_e), __FILE__, __LINE__); \
    return; } } while (0)

namespace ag {
#define LAS __attribute__((address_space(3)))
typedef unsigned short bf16_t;
typedef short bf16x8 __attribute__((ext_vector_type(8)));
typedef short bf16x4 __attribute__((ext_vector_type(4)));
typedef float f32x4 __attribute__((ext_vector_type(4)));
typedef unsigned u32x4 __attribute__((ext_vector_type(4)));
typedef unsigned u32x2 __attribute__((ext_vector_type(2)));

constexpr int NRANK = 8, SH_ROWS = 512;
constexpr int GM = 4096, GK = 4096, GN = 4096;
constexpr int BM = 256, BK = 64, HALF = 128, HTB = HALF * BK * 2, STAGE_BYTES = 8 * HTB, NXCD = 8, WGM = 8;
constexpr int NT = GK / BK, NWG = (GM / BM) * (GN / BM);
constexpr int KC = 256, NCH = GK / KC, DLEAD = 2;
constexpr size_t WS_CTR = 0, WS_TMO = 128, WS_FLAGS = 4096, WS_X = 1u << 20, XBYTES = (size_t)GM * GK * 2;
constexpr unsigned long long DEADLINE_TICKS = 20000000ull;
constexpr int LDS_BYTES = STAGE_BYTES + 64;

struct Params { const bf16_t* x; const bf16_t* w; bf16_t* out; unsigned char* ws[NRANK]; int rank; };

__device__ __forceinline__ int lds_byte(int r, int c) { const int st = (r >> 4) * 2 + (c >> 5), rr = r & 15, cc = c & 31, ob = rr * 64 + cc * 2; return st * 1024 + (ob ^ (((ob >> 9) & 1) << 5)); }
__device__ __forceinline__ void stage_rc(int b, int& R, int& C) { const int st = b / 1024, sb = b % 1024, swz = sb ^ (((sb >> 9) & 1) << 5); R = (st >> 1) * 16 + swz / 64; C = (st & 1) * 32 + (swz % 64) / 2; }
__device__ __forceinline__ unsigned cvt_pk_bf16(float lo, float hi) { unsigned r; asm volatile("v_cvt_pk_bf16_f32 %0, %1, %2" : "=v"(r) : "v"(lo), "v"(hi)); return r; }

__global__ __launch_bounds__(512, 2) void ag_gemm(const Params P) {
    extern __shared__ __attribute__((aligned(16))) unsigned char lds_raw[];
    LAS unsigned char* lds = (LAS unsigned char*)lds_raw;
    const int tid = threadIdx.x, wid = __builtin_amdgcn_readfirstlane(tid >> 6), lane = tid & 63, wr = wid >> 2, wc = wid & 3, fr = lane & 15, fq = lane >> 4;
    const int rank = P.rank;
    unsigned char* const wsl = P.ws[rank];
    unsigned* const tmo = (unsigned*)(wsl + WS_TMO);
    unsigned ticket = 0;
    if (tid == 0) ticket = __hip_atomic_fetch_add((unsigned*)(wsl + WS_CTR), 1u, __ATOMIC_RELAXED, __HIP_MEMORY_SCOPE_AGENT);
    unsigned call = 0; size_t xoff = 0;
    int pm, pn;
    { const int nN = GN / BM; int wgid = blockIdx.x; const int q = NWG / NXCD, xcd = wgid % NXCD, off = wgid / NXCD; wgid = xcd * q + off;
      const int nig = WGM * nN; pm = (wgid / nig) * WGM + (wgid % nig) % WGM; pn = (wgid % nig) / WGM; }
    const int src = pm >> 1; const bool local = (src == rank);
    unsigned voffA, voffB;
    { int R, C; stage_rc((tid & 255) * 16, R, C); voffA = (unsigned)(R * GK + C) * 2u;
      const int b = (tid & 255) * 16, row = b >> 8, segp = (b & 255) >> 5, f = (row & 3) | (((row >> 3) & 1) << 2), col = ((segp ^ f) << 4) + ((b >> 4) & 1) * 8;
      voffB = (unsigned)(row * GN + col) * 2u; }
    const size_t kstepA = (size_t)(BK * 2), hstepA = (size_t)HALF * GK * 2, pstepA = (size_t)32 * GK * 2;
    const size_t kstepB = (size_t)BK * GN * 2, hstepB = (size_t)HALF * 2, pstepB = (size_t)16 * GN * 2;
    const unsigned ldsw = (unsigned)(wid & 3) * 1024u;
    const int aoff = lds_byte(wr * 64 + fr, fq * 8);
    unsigned baddr[2];
    { const int g = fq, q = (lane & 15) >> 2, p = lane & 3, f = q | ((g & 1) << 2);
#pragma unroll
      for (int n = 0; n < 2; ++n) baddr[n] = (unsigned)(size_t)lds + 4 * HTB + 256 * (8 * g + q) + 32 * ((wc * 2 + n) ^ f) + 8 * p; }
#define SA(b, h) (((b) * 2 + (h)) * HTB)
#define SB(b, h) ((4 + (b) * 2 + (h)) * HTB)
#define STAGE(bufoff, gbase, voff, pstep) do { if (wr == 0) { _Pragma("unroll") for (int _i = 0; _i < 4; ++_i) \
        __builtin_amdgcn_global_load_lds((const unsigned*)((const char*)(gbase) + _i * (pstep) + (voff)), (LAS unsigned*)(lds + (bufoff) + ldsw + _i * 4096), 16, 0, 0); } } while (0)
#define STA(bufoff, gbase) STAGE(bufoff, gbase, voffA, pstepA)
#define STB(bufoff, gbase) STAGE(bufoff, gbase, voffB, pstepB)
#define LDA(dst, b, h) do { _Pragma("unroll") for (int m = 0; m < 4; ++m) _Pragma("unroll") for (int k = 0; k < 2; ++k) dst[m][k] = *(const LAS bf16x8*)(lds + SA(b, h) + aoff + m * 2048 + k * 1024); } while (0)
#define TRRD(d, n, off) asm volatile("ds_read_b64_tr_b16 %0, %1 offset:%2" : "=v"(d) : "v"(baddr[n]), "i"(off))
#define LDB(dst, b, h) do { _Pragma("unroll") for (int n = 0; n < 2; ++n) _Pragma("unroll") for (int k = 0; k < 2; ++k) { \
        bf16x4 lo, hi; TRRD(lo, n, ((b) * 2 + (h)) * HTB + k * 8192); TRRD(hi, n, ((b) * 2 + (h)) * HTB + k * 8192 + 1024); \
        dst[n][k] = __builtin_shufflevector(lo, hi, 0, 1, 2, 3, 4, 5, 6, 7); } } while (0)
#define WAIT_LB(Bx) asm volatile("s_waitcnt lgkmcnt(0)" : "+v"(Bx[0][0]), "+v"(Bx[0][1]), "+v"(Bx[1][0]), "+v"(Bx[1][1]) :: "memory")
#define MMA(ai, bj, At, Bt) do { __builtin_amdgcn_s_setprio(1); _Pragma("unroll") for (int m = 0; m < 4; ++m) _Pragma("unroll") for (int n = 0; n < 2; ++n) _Pragma("unroll") for (int k = 0; k < 2; ++k) \
        acc[ai][bj][m][n] = __builtin_amdgcn_mfma_f32_16x16x32_bf16(Bt[n][k], At[m][k], acc[ai][bj][m][n], 0, 0, 0); __builtin_amdgcn_s_setprio(0); } while (0)
#define WAIT_V(n) asm volatile("s_waitcnt vmcnt(" #n ")" ::: "memory")
#define WAIT_L(n) asm volatile("s_waitcnt lgkmcnt(" #n ")" ::: "memory")
#define BAR __builtin_amdgcn_s_barrier()
#define SCHED __builtin_amdgcn_sched_barrier(0)

    unsigned wsv_lo, wsv_hi;
    { unsigned long long v = (unsigned long long)P.ws[0];
#pragma unroll
      for (int p = 1; p < NRANK; ++p) v = (lane & 7) == p ? (unsigned long long)P.ws[p] : v;
      wsv_lo = (unsigned)v; wsv_hi = (unsigned)(v >> 32); }
    const unsigned pvoff = (unsigned)((rank * SH_ROWS + 2 * blockIdx.x + (lane >> 5)) * GK + (lane & 31) * 8) * 2u;
    auto peer_k = [&](int k) -> int { return (rank + 1 + (k + (int)blockIdx.x) % (NRANK - 1)) & (NRANK - 1); };
    auto peer_ws = [&](int p) -> unsigned char* { return (unsigned char*)(((unsigned long long)(unsigned)__builtin_amdgcn_readlane((int)wsv_hi, p) << 32) | (unsigned)__builtin_amdgcn_readlane((int)wsv_lo, p)); };
    u32x4 piece = (u32x4){0u, 0u, 0u, 0u};
    auto load_piece = [&](u32x4& dst, int c) { const char* base = (const char*)P.x - (size_t)rank * SH_ROWS * GK * 2 + (size_t)c * KC * 2;
        asm volatile("global_load_dwordx4 %0, %1, %2" : "=v"(dst) : "v"(pvoff), "s"(base) : "memory"); };
    auto push_one = [&](int c, int k, u32x4 v) {
        const unsigned char* base = peer_ws(peer_k(k)) + xoff + (size_t)c * KC * 2;
        asm volatile("global_store_dwordx4 %0, %1, %2 sc0 sc1\n\ts_nop 1" :: "v"(pvoff), "v"(v), "s"(base) : "memory");
        };
    auto publish = [&](int c, bool drained = false) {
        if (!drained) asm volatile("s_waitcnt vmcnt(0)" ::: "memory");
        if (lane < NRANK && lane != rank) { const unsigned* fp = (const unsigned*)((((unsigned long long)wsv_hi << 32) | wsv_lo) + WS_FLAGS) + ((rank * NCH + c) * NWG + blockIdx.x);
            asm volatile("global_store_dword %0, %1, off sc0 sc1\n\ts_nop 1" :: "v"(fp), "v"(call) : "memory"); } };
    auto flag_ptr = [&](int c) -> const unsigned long long* { return (const unsigned long long*)((const unsigned*)(wsl + WS_FLAGS) + ((src * NCH + c) * NWG + (pm & 1) * 128)) + lane; };
    auto flags_ok = [&](unsigned long long v) -> bool { return __all((unsigned)v >= call && (unsigned)(v >> 32) >= call); };
    auto sys_load64 = [&](const unsigned long long* p) -> unsigned long long { unsigned long long v; asm volatile("global_load_dwordx2 %0, %1, off sc0 sc1\n\ts_waitcnt vmcnt(0)" : "=&v"(v) : "v"(p) : "memory"); return v; };
    auto tmo_load = [&]() -> unsigned { unsigned v; asm volatile("global_load_dword %0, %1, off sc1\n\ts_waitcnt vmcnt(0)" : "=&v"(v) : "v"(tmo) : "memory"); return __builtin_amdgcn_readfirstlane(v); };
    auto wait_chunk = [&](int c) {
        const unsigned long long* fl = flag_ptr(c);
        const unsigned long long t0 = __builtin_amdgcn_s_memrealtime();
        for (;;) {
            if (flags_ok(sys_load64(fl))) break;
            if (tmo_load() == call || __builtin_amdgcn_s_memrealtime() - t0 > DEADLINE_TICKS) {
                asm volatile("global_store_dword %0, %1, off sc1\n\ts_nop 1" :: "v"(tmo), "v"(call) : "memory");
                break; }
            __builtin_amdgcn_s_sleep(1);
        } };
    auto acquire = [&]() {
        asm volatile("buffer_inv sc0 sc1\n\ts_waitcnt vmcnt(0)" ::: "memory");
        };
    unsigned long long early = 0;
    auto poll_issue = [&](int c) { const unsigned long long* fl = flag_ptr(c); asm volatile("global_load_dwordx2 %0, %1, off sc0 sc1" : "=v"(early) : "v"(fl) : "memory"); };
    auto poll_check = [&](int c) { asm volatile("s_waitcnt vmcnt(0)" : "+v"(early) :: "memory");
        if (!flags_ok(early)) wait_chunk(c);
        };

    f32x4 acc[2][2][4][2];
#pragma unroll
    for (int a = 0; a < 2; ++a)
#pragma unroll
        for (int b = 0; b < 2; ++b)
#pragma unroll
            for (int m = 0; m < 4; ++m)
#pragma unroll
                for (int n = 0; n < 2; ++n) acc[a][b][m][n] = (f32x4){0.f, 0.f, 0.f, 0.f};
    bf16x8 At[4][2], B0[2][2], B1[2][2];
    unsigned tmo_v = 0;
    const char* cB = (const char*)P.w + (size_t)pn * BM * 2;
#define TP(acc, code) do { code; } while (0)
    STB(SB(0, 0), cB); STB(SB(0, 1), cB + hstepB);
    u32x4 p0, p1;
    if (wid == 4) { load_piece(p0, 0); load_piece(p1, 1); load_piece(piece, 2); }
    { LAS unsigned* bc = (LAS unsigned*)(lds + STAGE_BYTES);
      if (tid == 0) bc[0] = ticket / (unsigned)NWG + 1u;
      __syncthreads(); }
    call = __builtin_amdgcn_readfirstlane(*(LAS unsigned*)(lds + STAGE_BYTES));
    xoff = WS_X + (size_t)(call & 1u) * XBYTES;
    const char* cA = local ? (const char*)P.x + (size_t)(pm & 1) * BM * GK * 2 : (const char*)(wsl + xoff) + (size_t)pm * BM * GK * 2;
    if (wid == 5 && !local) { wait_chunk(0); acquire(); }
    if (wid == 4) {
        asm volatile("s_waitcnt vmcnt(0)" : "+v"(p0), "+v"(p1), "+v"(piece) :: "memory");
        for (int k = 0; k < NRANK - 1; ++k) push_one(0, k, p0);
        for (int k = 0; k < NRANK - 1; ++k) push_one(1, k, p1);
        publish(0);
        publish(1, true);
    }
    asm volatile("" ::: "memory"); BAR; asm volatile("" ::: "memory");
    STA(SA(0, 0), cA); STA(SA(0, 1), cA + hstepA);
    if (wr == 1) BAR;
    WAIT_V(4); BAR;
    STB(SB(1, 0), cB + kstepB); STA(SA(1, 0), cA + kstepA); STB(SB(1, 1), cB + hstepB + kstepB);
    WAIT_V(12); BAR;
    for (int t = 0; t < NT; t += 2) {
        const bool last = (t == NT - 2);
        const int t2 = last ? 0 : t + 2;
        const char* a1 = cA + (size_t)(t + 1) * kstepA;
        const char* a2 = cA + (size_t)t2 * kstepA; const char* b2 = cB + (size_t)t2 * kstepB;
        const char* a3 = a2 + kstepA; const char* b3 = b2 + kstepB;
        const int it = t >> 1, cp = (it >> 1) + DLEAD, cn = (it >> 1) + 1;
        const bool cw = (wid == 4), ev = (it & 1) == 0, dopush = cw && ev && cp < NCH;
#define PUSH(k) do { if (dopush) TP(tp_push, push_one(cp, k, piece)); } while (0)
        if (last) asm volatile("global_load_dword %0, %1, off sc1" : "=v"(tmo_v) : "v"(tmo) : "memory");
        if (cw && !ev && cn < NCH && !local) TP(tp_wait, poll_check(cn));
        if (cw && ev) asm volatile("s_waitcnt vmcnt(0)" : "+v"(piece) :: "memory");
        PUSH(0);
        LDB(B0, 0, 0); SCHED; LDA(At, 0, 0); STA(SA(1, 1), a1 + hstepA);
        WAIT_L(8); BAR; WAIT_LB(B0); MMA(0, 0, At, B0); BAR; SCHED;
        PUSH(1);
        LDB(B1, 0, 1); STB(SB(0, 0), b2);
        BAR; WAIT_LB(B1); MMA(0, 1, At, B1); BAR;
        PUSH(2);
        LDA(At, 0, 1); STA(SA(0, 0), a2);
        BAR; WAIT_L(0); MMA(1, 0, At, B0); BAR; SCHED;
        PUSH(3);
        STB(SB(0, 1), b2 + hstepB);
        WAIT_V(12); BAR; MMA(1, 1, At, B1); BAR;
        PUSH(4);
        if (cw && !ev && cp < NCH) TP(tp_pub, publish(cp));
        LDB(B0, 1, 0); SCHED; LDA(At, 1, 0); STA(SA(0, 1), a2 + hstepA);
        WAIT_L(8); BAR; WAIT_LB(B0); MMA(0, 0, At, B0); BAR; SCHED;
        PUSH(5);
        LDB(B1, 1, 1); STB(SB(1, 0), b3);
        BAR; WAIT_LB(B1); MMA(0, 1, At, B1); BAR;
        PUSH(6);
        LDA(At, 1, 1); STA(SA(1, 0), a3);
        BAR; WAIT_L(0); MMA(1, 0, At, B0); BAR; SCHED;
        if (cw && ev && cp + 1 < NCH) load_piece(piece, cp + 1);
        if (cw && ev && cn < NCH && !local) poll_issue(cn);
        STB(SB(1, 1), b3 + hstepB);
        WAIT_V(12); BAR; MMA(1, 1, At, B1); BAR;
    }
    asm volatile("s_waitcnt vmcnt(0)" : "+v"(tmo_v) :: "memory");
    if (wr == 0) BAR;
    const bool dead = __builtin_amdgcn_readfirstlane(tmo_v) == call;
    const int row0 = pm * BM + wr * 64 + fr, col0 = pn * BM + wc * 32 + (fq & 1) * 16 + (fq >> 1) * 8;
#pragma unroll
    for (int ai = 0; ai < 2; ++ai)
#pragma unroll
        for (int m = 0; m < 4; ++m) { bf16_t* rowp = P.out + (size_t)(row0 + ai * HALF + m * 16) * GN + col0;
#pragma unroll
            for (int bj = 0; bj < 2; ++bj) { f32x4 v0 = acc[ai][bj][m][0], v1 = acc[ai][bj][m][1];
#pragma unroll
                for (int j = 0; j < 4; ++j) { v0[j] = fmaxf(v0[j], 0.f); v1[j] = fmaxf(v1[j], 0.f); }
                const unsigned a0 = cvt_pk_bf16(v0[0], v0[1]), a1 = cvt_pk_bf16(v0[2], v0[3]), b0 = cvt_pk_bf16(v1[0], v1[1]), b1 = cvt_pk_bf16(v1[2], v1[3]);
                const auto s0 = __builtin_amdgcn_permlane16_swap(a0, b0, false, false), s1 = __builtin_amdgcn_permlane16_swap(a1, b1, false, false);
                u32x4 w; w.x = s0[0]; w.y = s1[0]; w.z = s0[1]; w.w = s1[1];
                if (dead) w = (u32x4){0x7fc07fc0u, 0x7fc07fc0u, 0x7fc07fc0u, 0x7fc07fc0u};
                *(u32x4*)(rowp + bj * HALF) = w; } }
}
}

extern "C" void kernel_launch(int rank, int nranks, ncclComm_t comm,
                              hipStream_t stream,
                              void* const* d_in, const int* in_sizes, int n_in,
                              void* d_out, int out_size,
                              const int* in_dtypes, int out_dtype,
                              void* const* d_out_all,
                              void* const* d_ws_all, size_t ws_bytes) {
    if (nranks != ag::NRANK || (unsigned)rank >= (unsigned)nranks || ws_bytes < ag::WS_X + 2 * ag::XBYTES) { fprintf(stderr, "unexpected problem shape\n"); return; }
    HIP_CHECK(hipFuncSetAttribute((const void*)ag::ag_gemm, hipFuncAttributeMaxDynamicSharedMemorySize, ag::LDS_BYTES));
    ag::Params P;
    P.x = (const ag::bf16_t*)d_in[0]; P.w = (const ag::bf16_t*)d_in[1]; P.out = (ag::bf16_t*)d_out; P.rank = rank;
    for (int p = 0; p < ag::NRANK; ++p) P.ws[p] = (unsigned char*)d_ws_all[p];
    ag::ag_gemm<<<ag::NWG, 512, ag::LDS_BYTES, stream>>>(P);
}
```
